```python
import numpy as np

kernel_name = "gpu_gemm_ar_m4096_k4096_n4096_f32_none"


ABSMAX_RTOL = 0.001

COMPUTE_DTYPE = "float32"
EPILOGUE = "none"
IN_QUANT = None
OUT_QUANT = None
IN_DTYPES = {"x": "float32", "w_mat": "float32"}
WORLD_SIZE = 8

def setup_inputs():

    x = np.random.randn(4096, 4096).astype(np.float32)
    w_mat = (np.random.randn(4096, 4096) / np.sqrt(4096)).astype(np.float32)
    return {"x": x, "w_mat": w_mat}

def _epilogue(y):
    return y

def reference(x, w_mat):
    y = (x @ w_mat).astype(np.float32)
    return _epilogue(y)
```

```cpp
#include <hip/hip_runtime.h>
#include <rccl/rccl.h>
#include <cstdio>

#define HIP_CHECK(expr) do { hipError_t _e = (expr); if (_e != hipSuccess) { \
    fprintf(stderr, "HIP error %s at %s:%d\n", hipGetErrorString(_e), __FILE__, __LINE__); \
    return; } } while (0)

typedef float f32x4 __attribute__((ext_vector_type(4)));
typedef float f32x16 __attribute__((ext_vector_type(16)));
typedef _Float16 h16x8 __attribute__((ext_vector_type(8)));
typedef unsigned u32x4 __attribute__((ext_vector_type(4)));
#define LAS __attribute__((address_space(3)))
#define GAS __attribute__((address_space(1)))

constexpr int NR = 8;
constexpr int MD = 4096, ND = 4096;
constexpr int KL = 512;
constexpr int KS = 64;
constexpr int GRID = 256, BLOCK = 512;
constexpr int NCOMM = 6;
#define CW 4
constexpr int NUNIT = 2048;
constexpr int XPAN = 13, WPAN = 16;

constexpr unsigned OFF_CTR = 0;
constexpr unsigned OFF_TMO = 32768;
constexpr unsigned OFF_FX = 40960;
constexpr unsigned OFF_FW = 45056;
constexpr unsigned OFF_DATA = 1u << 20;
constexpr unsigned HALF_BYTES = 32u << 20;
constexpr unsigned PAR_BYTES = 64u << 20;
constexpr unsigned WS_USED = OFF_DATA + 2 * PAR_BYTES;
constexpr unsigned OFF_DBG = WS_USED;
constexpr unsigned STEP_BYTES = 512u << 10;

constexpr int A_BYTES = 40960, B_BYTES = 32768, STAGE = A_BYTES + B_BYTES;
constexpr int LDS_BYTES = 2 * STAGE + 64;
constexpr unsigned long long DEADLINE_TICKS = 50000000ull;
constexpr int AUX_SYS = 17;

#define WAIT_V(n) asm volatile("s_waitcnt vmcnt(%0)" :: "n"(n) : "memory")

#define STAMP(idx) do {} while (0)

__device__ __forceinline__ u32x4 cvt8(float a, float b, float c, float d, float e, float f, float g, float h) {
  h16x8 r;
  r[0] = (_Float16)a; r[1] = (_Float16)b; r[2] = (_Float16)c; r[3] = (_Float16)d;
  r[4] = (_Float16)e; r[5] = (_Float16)f; r[6] = (_Float16)g; r[7] = (_Float16)h;
  return __builtin_bit_cast(u32x4, r);
}

struct UnitDesc { unsigned isx, lsoff, ssoff, flag; };

__global__ void __launch_bounds__(BLOCK, 2)
fused_gemm_ar(const float* __restrict__ xs, const float* __restrict__ wsrc, float* __restrict__ out,
              void* const* __restrict__ ws_all, int rank) {
  extern __shared__ __attribute__((aligned(1024))) char smem[];
  const unsigned lane = threadIdx.x & 63;
  const unsigned w = __builtin_amdgcn_readfirstlane(threadIdx.x >> 6);
  const unsigned g = blockIdx.x & 7, q = blockIdx.x >> 3;
  GAS char* ws = (GAS char*)ws_all[rank];

  if (q < NCOMM) {
    if (w >= CW) return;
    STAMP(0);
    unsigned call = 0;
    if (lane == 0)
      call = __hip_atomic_fetch_add((GAS unsigned*)(ws + OFF_CTR + blockIdx.x * 128 + w * 4), 1u, __ATOMIC_RELAXED, __HIP_MEMORY_SCOPE_AGENT) + 1;
    const unsigned p = ((unsigned)rank + 1 + g) & (NR - 1);
    GAS char* dws = (GAS char*)ws_all[p];
    const auto rd = __builtin_amdgcn_make_buffer_rsrc((void*)dws, 0, WS_USED, 0x00020000);
    const unsigned lvx = (lane & 31) * (KL * 4) + (lane >> 5) * 128;
    const unsigned lvw = ((lane >> 4 & 1) * 32 + (lane >> 5)) * (ND * 4) + (lane & 15) * 16;
    const unsigned svx = lane * 16;
    const unsigned svw = (lane >> 5) * 2048 + ((lane >> 4 & 1) * 32 + (lane & 15)) * 16;
    const unsigned NW = NCOMM * CW, lw = q * CW + w;
    const unsigned rot = g * 36;
    auto desc = [&](unsigned n) {
      UnitDesc d;
      const unsigned j = n >> 8, r = (n + rot) & 255, b = r >> 4, e = r & 15, t = (unsigned)rank * 8 + j;
      d.isx = e >> 3;
      const unsigned rb = 8 * b + (e & 7), cg = 4 * b + (e >> 1), u = e & 1;
      d.lsoff = d.isx ? rb * 32 * (KL * 4) + j * 256 : (j * 64 + 2 * u) * (ND * 4) + cg * 256;
      d.ssoff = t * STEP_BYTES + (d.isx ? rb * 4096 : HALF_BYTES + cg * 8192 + u * 4096);
      d.flag = (d.isx ? OFF_FX + (rb / 10) * 256 : OFF_FW + b * 256) + t * 4;
      if (n >= NUNIT) { d.lsoff = 0x40000000u; d.ssoff = 0x40000000u; }
      return d;
    };
    static_assert(MD * KL == KL * ND, "one record count serves both shards");
    auto load = [&](f32x4 (&v)[8], const UnitDesc& d) {
      const unsigned long long base = (unsigned long long)(d.isx ? xs : wsrc);
      const u32x4 rs = {(unsigned)base, (unsigned)(base >> 32) & 0xffffu, (unsigned)(MD * KL * 4), 0x00020000u};
      const unsigned lv = d.isx ? lvx : lvw, st = d.isx ? 16u : 4u * ND * 4u;
#pragma unroll
      for (int i = 0; i < 8; ++i)
        asm volatile("buffer_load_dwordx4 %0, %1, %2, %3 offen" : "=v"(v[i]) : "v"(lv), "s"(rs), "s"(d.lsoff + i * st) : "memory");
    };
#define WAIT_LD(v, n) asm volatile("s_waitcnt vmcnt(%8)" : "+v"(v[0]), "+v"(v[1]), "+v"(v[2]), "+v"(v[3]), "+v"(v[4]), "+v"(v[5]), "+v"(v[6]), "+v"(v[7]) : "n"(n) : "memory")
    unsigned dbase = 0;
    auto store = [&](const f32x4 (&v)[8], const UnitDesc& d) {
      const unsigned sv = d.isx ? svx : svw;
      const unsigned s2 = d.isx ? 2048u : 256u;
#pragma unroll
      for (int e = 0; e < 4; ++e) {
        const u32x4 o = cvt8(v[0][e], v[1][e], v[2][e], v[3][e], v[4][e], v[5][e], v[6][e], v[7][e]);
        __builtin_amdgcn_raw_buffer_store_b128(o, rd, sv + (e & 1) * 1024, dbase + d.ssoff + (e >> 1) * s2, AUX_SYS);
      }
    };
    auto signal = [&](const UnitDesc& d, bool valid) {
      if (valid && lane == 0)
        __hip_atomic_fetch_add((GAS unsigned*)(dws + d.flag), 1u, __ATOMIC_RELAXED, __HIP_MEMORY_SCOPE_SYSTEM);
    };
    f32x4 va[8], vb[8];
    UnitDesc da = desc(lw), db, dprev = da;
    load(va, da);
    call = __builtin_amdgcn_readfirstlane(call);
    dbase = OFF_DATA + (call & 1) * PAR_BYTES;
    bool have_prev = false;
    for (unsigned n = lw; n < NUNIT; n += 2 * NW) {
      db = desc(n + NW);
      load(vb, db);
      if (!have_prev) WAIT_V(8);
      WAIT_LD(va, 12);
      store(va, da);
      WAIT_V(12);
      signal(dprev, have_prev);
      dprev = da;
      da = desc(n + 2 * NW);
      load(va, da);
      WAIT_LD(vb, 12);
      store(vb, db);
      WAIT_V(12);
      signal(dprev, true);
      dprev = db; have_prev = (n + NW < NUNIT);
    }
    WAIT_V(0);
    signal(dprev, have_prev);
    STAMP(1);
    return;
  }

  STAMP(0);
  const unsigned L = (g & 1) * 26 + (q - NCOMM);
  const unsigned pi = L >> 2, pj = (g >> 1) * 4 + (L & 3);
  const unsigned wr = w >> 2, wc = w & 3;
  volatile LAS unsigned* sel = (volatile LAS unsigned*)(smem + 2 * STAGE);

  unsigned long long remaining = ~0ull, ready = 0;
  unsigned call = 0, tx = 0, tw = 0;
  const GAS unsigned* fx = (const GAS unsigned*)(ws + OFF_FX) + pi * 64 + lane;
  const GAS unsigned* fw = (const GAS unsigned*)(ws + OFF_FW) + pj * 64 + lane;
  GAS unsigned* tmo = (GAS unsigned*)(ws + OFF_TMO);

  auto pick = [&]() -> unsigned {
    if (!(ready & remaining)) {
      const unsigned long long t0 = __builtin_amdgcn_s_memrealtime();
      for (;;) {
        const unsigned a = __hip_atomic_load(fx, __ATOMIC_RELAXED, __HIP_MEMORY_SCOPE_SYSTEM);
        const unsigned b = __hip_atomic_load(fw, __ATOMIC_RELAXED, __HIP_MEMORY_SCOPE_SYSTEM);
        ready = __ballot(a >= tx && b >= tw);
        if (ready & remaining) break;
        const unsigned tm = __builtin_amdgcn_readfirstlane(__hip_atomic_load(tmo, __ATOMIC_RELAXED, __HIP_MEMORY_SCOPE_AGENT));
        if (tm == call || __builtin_amdgcn_s_memrealtime() - t0 > DEADLINE_TICKS) {
          if (lane == 0) __hip_atomic_store(tmo, call, __ATOMIC_RELAXED, __HIP_MEMORY_SCOPE_AGENT);
          return 0x100u;
        }
        __builtin_amdgcn_s_sleep(2);
      }
    }
    const unsigned k = __builtin_ctzll(ready & remaining);
    remaining &= ~(1ull << k);
    return k;
  };

  if (w == 0) {
    unsigned c = 0;
    if (lane == 0)
      c = __hip_atomic_fetch_add((GAS unsigned*)(ws + OFF_CTR + blockIdx.x * 128), 1u, __ATOMIC_RELAXED, __HIP_MEMORY_SCOPE_AGENT) + 1;
    call = __builtin_amdgcn_readfirstlane(c);
    tx = call * (pi == XPAN - 1 ? 8u : 10u);
    tw = call * 8u;
    const unsigned k = pick();
    if (lane == 0) { sel[0] = k; sel[2] = call; }
  }
  asm volatile("s_waitcnt lgkmcnt(0)" ::: "memory");
  __builtin_amdgcn_s_barrier();
  asm volatile("" ::: "memory");
  const unsigned par = __builtin_amdgcn_readfirstlane(sel[2]) & 1;
  const GAS char* ga = ws + OFF_DATA + par * PAR_BYTES + pi * A_BYTES + w * 5120 + 2048 + lane * 16;
  const GAS char* gb = ws + OFF_DATA + par * PAR_BYTES + HALF_BYTES + pj * B_BYTES + w * 4096 + 2048 + lane * 16;
  const int na = (pi == XPAN - 1) ? (int)32 - (int)w * 5 : 5;
  auto stage = [&](unsigned k, unsigned buf) {
    const GAS char* a = ga + (size_t)k * STEP_BYTES;
    const GAS char* b = gb + (size_t)k * STEP_BYTES;
    LAS char* la = (LAS char*)smem + buf * STAGE + w * 5120 + 2048;
    LAS char* lb = (LAS char*)smem + buf * STAGE + A_BYTES + w * 4096 + 2048;
    __builtin_amdgcn_global_load_lds(b, lb, 16, -2048, AUX_SYS);
    __builtin_amdgcn_global_load_lds(b, lb, 16, -1024, AUX_SYS);
    __builtin_amdgcn_global_load_lds(b, lb, 16, 0, AUX_SYS);
    __builtin_amdgcn_global_load_lds(b, lb, 16, 1024, AUX_SYS);
    if (na > 0) __builtin_amdgcn_global_load_lds(a, la, 16, -2048, AUX_SYS);
    if (na > 1) __builtin_amdgcn_global_load_lds(a, la, 16, -1024, AUX_SYS);
    if (na > 2) {
      __builtin_amdgcn_global_load_lds(a, la, 16, 0, AUX_SYS);
      __builtin_amdgcn_global_load_lds(a, la, 16, 1024, AUX_SYS);
      __builtin_amdgcn_global_load_lds(a, la, 16, 2048, AUX_SYS);
    }
  };

  f32x16 acc[5][2];
#pragma unroll
  for (int m = 0; m < 5; ++m)
#pragma unroll
    for (int n = 0; n < 2; ++n)
#pragma unroll
      for (int r = 0; r < 16; ++r) acc[m][n][r] = 0.f;

  unsigned bad = 0;
  {
    const unsigned k0 = __builtin_amdgcn_readfirstlane(sel[0]);
    bad = k0 >> 8;
    if (!bad) stage(k0, 0);
    if (w == 0) { const unsigned k = bad ? 0x100u : pick(); if (lane == 0) sel[1] = k; }
    asm volatile("s_waitcnt vmcnt(0) lgkmcnt(0)" ::: "memory");
    __builtin_amdgcn_s_barrier();
    asm volatile("" ::: "memory");
  }
  STAMP(1);
  const unsigned aoff = wr * 20480 + lane * 16, boff = A_BYTES + wc * 8192 + lane * 16;
  for (int t = 0; t < KS && !bad; ++t) {
    const unsigned cur = t & 1;
    unsigned pa = 0, pb = 0;
    if (t < KS - 1) {
      const unsigned kn = __builtin_amdgcn_readfirstlane(sel[cur ^ 1]);
      bad = kn >> 8;
      if (!bad) stage(kn, cur ^ 1);
      if (w == 0) {
        if (t < KS - 2) {
          asm volatile("global_load_dword %0, %1, off sc0 sc1" : "=v"(pa) : "v"(fx) : "memory");
          asm volatile("global_load_dword %0, %1, off sc0 sc1" : "=v"(pb) : "v"(fw) : "memory");
        } else {
          asm volatile("global_load_dword %0, %1, off sc1" : "=v"(pa) : "v"(tmo) : "memory");
        }
      }
    } else {
      bad = __builtin_amdgcn_readfirstlane(sel[cur ^ 1]) >> 8;
    }
    const LAS char* sa = (const LAS char*)smem + cur * STAGE + aoff;
    const LAS char* sb = (const LAS char*)smem + cur * STAGE + boff;
#pragma unroll
    for (int s = 0; s < 4; ++s) {
      h16x8 bf[2], af[5];
#pragma unroll
      for (int n = 0; n < 2; ++n) bf[n] = *(const LAS h16x8*)(sb + s * 2048 + n * 1024);
#pragma unroll
      for (int m = 0; m < 5; ++m) af[m] = *(const LAS h16x8*)(sa + m * 4096 + s * 1024);
#pragma unroll
      for (int m = 0; m < 5; ++m)
#pragma unroll
        for (int n = 0; n < 2; ++n)
          acc[m][n] = __builtin_amdgcn_mfma_f32_32x32x16_f16(af[m], bf[n], acc[m][n], 0, 0, 0);
    }
    asm volatile("s_waitcnt vmcnt(0)" : "+v"(pa), "+v"(pb) :: "memory");
    if (w == 0 && t < KS - 1) {
      unsigned k;
      if (t < KS - 2) {
        ready = __ballot(pa >= tx && pb >= tw);
        k = bad ? 0x100u : pick();
      } else {
        k = (bad || __builtin_amdgcn_readfirstlane(pa) == call) ? 0x100u : 0u;
      }
      if (lane == 0) sel[cur] = k;
    }
    asm volatile("s_waitcnt lgkmcnt(0)" ::: "memory");
    __builtin_amdgcn_s_barrier();
    asm volatile("" ::: "memory");
    STAMP(2 + t);
  }

  const unsigned row0 = pi * 320 + wr * 160 + (lane >> 5) * 4 + (lane >> 4 & 1);
  float* op = out + (size_t)row0 * ND + pj * 256 + wc * 64 + (lane & 15) * 4;
  const float nanv = __builtin_nanf("");
#pragma unroll
  for (int m = 0; m < 5; ++m) {
    if (pi * 320 + wr * 160 + m * 32 < MD) {
#pragma unroll
      for (int r = 0; r < 16; r += 2) {
        const float a0 = acc[m][0][r], b0 = acc[m][0][r + 1], a1 = acc[m][1][r], b1 = acc[m][1][r + 1];
        const auto s0 = __builtin_amdgcn_permlane16_swap(__float_as_uint(a0), __float_as_uint(b0), false, false);
        const auto s1 = __builtin_amdgcn_permlane16_swap(__float_as_uint(a1), __float_as_uint(b1), false, false);
        const unsigned c0 = s0[0], c1 = s1[0], c2 = s0[1], c3 = s1[1];
        f32x4 o;
        o[0] = __uint_as_float(c0); o[1] = __uint_as_float(c1); o[2] = __uint_as_float(c2); o[3] = __uint_as_float(c3);
        if (bad) o = (f32x4){nanv, nanv, nanv, nanv};
        *(f32x4*)(op + (size_t)(m * 32 + (r & 3) + 8 * (r >> 2)) * ND) = o;
      }
    }
  }
}

extern "C" void kernel_launch(int rank, int nranks, ncclComm_t comm,
                              hipStream_t stream,
                              void* const* d_in, const int* in_sizes, int n_in,
                              void* d_out, int out_size,
                              const int* in_dtypes, int out_dtype,
                              void* const* d_out_all,
                              void* const* d_ws_all, size_t ws_bytes) {
  if (nranks != NR || (unsigned)rank >= (unsigned)NR || n_in != 2 || in_sizes[0] != MD * KL || in_sizes[1] != KL * ND ||
      out_size != MD * ND || ws_bytes < (size_t)WS_USED + (2u << 20)) {
    fprintf(stderr, "unexpected problem shape\n");
    return;
  }
  HIP_CHECK(hipFuncSetAttribute((const void*)fused_gemm_ar, hipFuncAttributeMaxDynamicSharedMemorySize, LDS_BYTES));
  fused_gemm_ar<<<GRID, BLOCK, LDS_BYTES, stream>>>((const float*)d_in[0], (const float*)d_in[1], (float*)d_out, d_ws_all, rank);
  HIP_CHECK(hipGetLastError());
}
```

```cpp
#include <hip/hip_runtime.h>
#include <rccl/rccl.h>
#include <cstddef>
#include <cstdio>

#define HIP_CHECK(expr) do { hipError_t _e = (expr); if (_e != hipSuccess) { \
    fprintf(stderr, "HIP error %s at %s:%d\n", hipGetErrorString(_e), __FILE__, __LINE__); \
    return; } } while (0)

typedef float f32x4 __attribute__((ext_vector_type(4)));
typedef _Float16 h16x8 __attribute__((ext_vector_type(8)));
typedef unsigned u32x4 __attribute__((ext_vector_type(4)));
#define LAS __attribute__((address_space(3)))
#define GAS __attribute__((address_space(1)))
#define CAS __attribute__((address_space(4)))

constexpr int NR = 8;
constexpr int MD = 4096, ND = 4096;
constexpr int KL = 512;
constexpr int KS = 64;
constexpr int GRID = 256, BLOCK = 512;
constexpr int NCOMM = 6;
constexpr int CW = 4;
constexpr int PUSHN = 1;
constexpr int XPAN = 13, WPAN = 16;
constexpr int QG = 8;
constexpr int NBLK = 8 * (XPAN - 1 + WPAN) - NCOMM * PUSHN;
constexpr int NSTEP = 1024 - 40 - 4 * NCOMM * PUSHN;
constexpr int OSTEP = 640;
constexpr unsigned CLIQUE = 0x4du;

constexpr unsigned OFF_CTR = 0;
constexpr unsigned OFF_TMO = 32768;
constexpr unsigned OFF_RDY = 49152;
constexpr unsigned OFF_DONE = 40960;
constexpr unsigned OFF_TKT = 61440;
constexpr unsigned FSTR = 4352;
constexpr unsigned OFF_FLG = 65536;
constexpr unsigned OFF_CNT = OFF_FLG + 262144;
__device__ __forceinline__ unsigned flag_off(unsigned row, unsigned t, unsigned call) { return OFF_FLG + row * FSTR + (call & 1) * 512 + t * 8; }
__device__ __forceinline__ unsigned tag_of(unsigned call) { return (call >> 1) % 255u + 1; }
constexpr unsigned OFF_DATA = 1u << 20;
constexpr unsigned HALF_BYTES = 32u << 20;
constexpr unsigned PAR_BYTES = 64u << 20;
constexpr unsigned WS_USED = OFF_DATA + 2 * PAR_BYTES;
constexpr unsigned STEP_BYTES = 512u << 10;
constexpr unsigned OOB = 0x40000000u;

constexpr int A_BYTES = 40960, B_BYTES = 32768, STAGE = A_BYTES + B_BYTES;
constexpr int LDS_BYTES = 2 * STAGE + 64;
constexpr unsigned RING = 8192, NSLOT = 16, AHEAD = 4;
static_assert(RING + NSLOT * 8192 <= LDS_BYTES && 256 + 4 * NSTEP <= RING, "link LDS layout");
constexpr unsigned long long DEADLINE_TICKS = 50000000ull;
constexpr int AUX_DMA = 17;
constexpr unsigned DIRECT = 160;
constexpr unsigned DRIFT = 2;
constexpr int PUSH_PACE = 2;
constexpr int STAGE_DELAY = 48;
constexpr int POLL_SLEEP = 4;
#define SYS __ATOMIC_RELAXED, __HIP_MEMORY_SCOPE_SYSTEM
#define AGT __ATOMIC_RELAXED, __HIP_MEMORY_SCOPE_AGENT

struct Params {
  const float* x; const float* w; float* out;
  char* ws[NR];
  char* outs[NR];
  int rank, pad;
  unsigned char order[NR - 1][256];
};

#define WAIT_V(n) asm volatile("s_waitcnt vmcnt(%0)" :: "n"(n) : "memory")
#define WAIT_L0 asm volatile("s_waitcnt lgkmcnt(0)" ::: "memory")
#define BARRIER do { asm volatile("" ::: "memory"); __builtin_amdgcn_s_barrier(); asm volatile("" ::: "memory"); } while (0)
#define ST128(data, rsrc, voff, soff, imm, bits) \
  asm volatile("buffer_store_dwordx4 %0, %1, %2, %3 offen offset:%4 " bits "\n\ts_nop 1" :: "v"(data), "v"(voff), "s"(rsrc), "s"(soff), "n"(imm) : "memory")
__device__ __forceinline__ u32x4 make_rsrc(const void* base, unsigned bytes) {
  const unsigned long long b = (unsigned long long)base;
  return (u32x4){(unsigned)b, (unsigned)(b >> 32) & 0xffffu, bytes, 0x00020000u};
}

__device__ __forceinline__ u32x4 cvt8(float a, float b, float c, float d, float e, float f, float g, float h) {
  h16x8 r;
  r[0] = (_Float16)a; r[1] = (_Float16)b; r[2] = (_Float16)c; r[3] = (_Float16)d;
  r[4] = (_Float16)e; r[5] = (_Float16)f; r[6] = (_Float16)g; r[7] = (_Float16)h;
  return __builtin_bit_cast(u32x4, r);
}

template <int N>
__device__ __forceinline__ void unit_load(f32x4 (&v)[N], const Params& P, unsigned isx, unsigned j, unsigned b, unsigned u, unsigned lane) {
  static_assert(MD * KL == KL * ND, "one record count serves both shards");
  const u32x4 rs = make_rsrc(isx ? P.x : P.w, MD * KL * 4);
  const unsigned lx = (lane >> 3) * (KL * 4) + ((lane & 7) ^ (lane >> 4)) * 16;
  const unsigned lv0 = isx ? lx : (lane >> 4) * 4 * (ND * 4) + (lane & 15) * 16, lv1 = isx ? lx ^ 64 : lv0;
  const unsigned so = isx ? (10 * b + u) * 32 * (KL * 4) + j * 256 : (j * 64 + 16 * (u & 1)) * (ND * 4) + (4 * b + (u >> 1)) * 256;
#pragma unroll
  for (int i = 0; i < 8; ++i) {
    const unsigned d = isx ? (i >> 1) * 8 * (KL * 4) + (i & 1) * 128 : (32 * (i >> 2) + (i & 3)) * (ND * 4);
    asm volatile("buffer_load_dwordx4 %0, %1, %2, %3 offen" : "=v"(v[i]) : "v"((i & 2) ? lv1 : lv0), "s"(rs), "s"(so + d) : "memory");
  }
  const unsigned se = isx ? (320 * b + 256 + 8 * u) * (KL * 4) + j * 256 : OOB;
#pragma unroll
  for (int i = 8; i < N; ++i)
    asm volatile("buffer_load_dwordx4 %0, %1, %2, %3 offen" : "=v"(v[i]) : "v"((u & 1) ? lv1 : lv0), "s"(rs), "s"(se + (i & 1) * 128) : "memory");
}
#define UNIT_WAIT(v, n) asm volatile("s_waitcnt vmcnt(%8)" : "+v"(v[0]), "+v"(v[1]), "+v"(v[2]), "+v"(v[3]), "+v"(v[4]), "+v"(v[5]), "+v"(v[6]), "+v"(v[7]) : "n"(n) : "memory")
template <int N>
__device__ __forceinline__ unsigned wave_amax(const f32x4 (&v)[N]) {
  unsigned m = 0;
#pragma unroll
  for (int i = 0; i < N; ++i)
#pragma unroll
    for (int e = 0; e < 4; ++e) { const float f = v[i][e]; m = max(m, __float_as_uint(f) & 0x7fffffffu); }
#define DPP_MAX(ctrl) m = max(m, (unsigned)__builtin_amdgcn_update_dpp(0, (int)m, ctrl, 0xf, 0xf, false))
  DPP_MAX(0x111); DPP_MAX(0x112); DPP_MAX(0x114); DPP_MAX(0x118); DPP_MAX(0x142); DPP_MAX(0x143);
  return __builtin_amdgcn_readlane((int)m, 63);
}
__device__ __forceinline__ int block_exp(unsigned amax) {
  const int a = (int)(amax >> 23) - 127;
  return (amax == 0 || (a >= -6 && a <= 14)) ? 0 : min(max(a & ~7, -24), 24);
}
__device__ __forceinline__ float pow2f(int e) { return __uint_as_float((unsigned)(127 + e) << 23); }
template <int PACE, int N>
__device__ __forceinline__ void unit_store(const f32x4 (&vr)[N], float sc, u32x4 rd, unsigned so, unsigned se, unsigned isx, unsigned lane) {
  const unsigned sv = lane * 16;
  f32x4 v[N];
#pragma unroll
  for (int i = 0; i < N; ++i) v[i] = vr[i] * sc;
#pragma unroll
  for (int e = 0; e < 4; ++e) {
    const u32x4 o = isx ? cvt8(v[2 * e][0], v[2 * e][1], v[2 * e][2], v[2 * e][3], v[2 * e + 1][0], v[2 * e + 1][1], v[2 * e + 1][2], v[2 * e + 1][3])
                        : cvt8(v[0][e], v[1][e], v[2][e], v[3][e], v[4][e], v[5][e], v[6][e], v[7][e]);
    if (e == 0) ST128(o, rd, sv, so, 0, "sc0 sc1"); else if (e == 1) ST128(o, rd, sv, so, 1024, "sc0 sc1");
    else if (e == 2) ST128(o, rd, sv, so, 2048, "sc0 sc1"); else ST128(o, rd, sv, so, 3072, "sc0 sc1");
    if (PACE > 0) WAIT_V(PACE - 1);
  }
  if (N > 8) {
    const u32x4 o = cvt8(v[N - 2][0], v[N - 2][1], v[N - 2][2], v[N - 2][3], v[N - 1][0], v[N - 1][1], v[N - 1][2], v[N - 1][3]);
    ST128(o, rd, sv, se, 0, "sc0 sc1");
  }
}
__host__ __device__ constexpr unsigned push_panel(unsigned link, unsigned i) { return (link * 2 + i) & 15; }

__global__ void __launch_bounds__(BLOCK, 2)
fused_gemm_ar(const Params P) {
  extern __shared__ __attribute__((aligned(1024))) char smem[];
  const unsigned lane = threadIdx.x & 63;
  const unsigned w = __builtin_amdgcn_readfirstlane(threadIdx.x >> 6);
  const unsigned g = blockIdx.x & 7, q = blockIdx.x >> 3;
  const unsigned rank = (unsigned)P.rank & (NR - 1);
  const CAS char* karg = (const CAS char*)__builtin_amdgcn_kernarg_segment_ptr();
  const CAS unsigned long long* wstab = (const CAS unsigned long long*)(karg + offsetof(Params, ws));
  GAS char* ws = (GAS char*)wstab[rank];
  GAS unsigned* tmo = (GAS unsigned*)(ws + OFF_TMO);
  const u32x4 rd = make_rsrc((const void*)ws, WS_USED);
#define CALL_ISSUE unsigned cv_; unsigned long long ex_; \
  asm volatile("s_mov_b64 %1, exec\n\ts_mov_b64 exec, 1\n\tglobal_atomic_add %0, %2, %3, off sc0 sc1\n\ts_mov_b64 exec, %1" \
               : "=v"(cv_), "=&s"(ex_) : "v"(ws + OFF_CTR + blockIdx.x * 128 + w * 4), "v"(1u) : "memory")
#define CALL_WAIT(n) ({ asm volatile("s_waitcnt vmcnt(%1)" : "+v"(cv_) : "n"(n) : "memory"); (unsigned)__builtin_amdgcn_readfirstlane(cv_) + 1; })

  unsigned p;
  {
    const unsigned mine = ((CLIQUE >> rank) & 1) ? CLIQUE : (~CLIQUE & 0xffu);
    unsigned m = (g < 4) ? mine : (~mine & 0xffu);
    for (unsigned i = 0; i < (g & 3); ++i) m &= m - 1;
    p = __builtin_ctz(m) & (NR - 1);
  }

  if (q < NCOMM && p != rank) {
    volatile LAS unsigned* ctl = (volatile LAS unsigned*)smem;
    LAS unsigned* tab = (LAS unsigned*)(smem + 256);
    GAS char* pws = (GAS char*)wstab[p];
    const unsigned dist = (p - rank) & 7, z0 = OSTEP * (dist - 1) / 7, ntot = NSTEP + OSTEP * dist / 7 - z0;
    const unsigned lv = lane * 16;
    const unsigned gate_off = lane < 8 ? OFF_RDY + (p * 8 + lane) * 128 : OFF_DONE + (p * 8 + (lane & 7)) * 128;
    const GAS unsigned* gate_word = (const GAS unsigned*)(ws + gate_off);
    const unsigned gate_mul = lane == 0 ? WPAN : 1;
    auto out_off = [&](unsigned z) -> unsigned {
      const unsigned tc = z / 40;
      return (rank * 320 + (z - tc * 40) * 8) * (ND * 4) + ((2 * (rank + 1) + tc) & 15) * 1024;
    };
    if (w >= CW) {
      const unsigned pb_ = push_panel((p - rank - 1) & 7, q);
      CALL_ISSUE;
      f32x4 va[8], vb[8];
      unit_load(va, P, 0, 0, pb_, (w - CW) * 2, lane);
      unit_load(vb, P, 0, 0, pb_, (w - CW) * 2 + 1, lane);
      BARRIER;
      const unsigned call = CALL_WAIT(16);
      const unsigned long long t0 = __builtin_amdgcn_s_memrealtime();
      {
        const u32x4 rp = make_rsrc((const void*)pws, WS_USED);
        const unsigned po = OFF_DATA + (call & 1) * PAR_BYTES + HALF_BYTES + rank * 8 * STEP_BYTES + pb_ * 32768 + (w - CW) * 8192;
        UNIT_WAIT(va, 8);
        unit_store<PUSH_PACE>(va, 1.f, rp, po, OOB, 0, lane);
        UNIT_WAIT(vb, 4);
        unit_store<PUSH_PACE>(vb, 1.f, rp, po + 4096, OOB, 0, lane);
        ctl[8 + w] = max(wave_amax(va), wave_amax(vb)) | 0x80000000u;
        u32x4 am;
        do am = *(volatile LAS u32x4*)(smem + 48);
        while (!((unsigned)__builtin_amdgcn_readfirstlane(am[0] & am[1] & am[2] & am[3]) >> 31) && __builtin_amdgcn_s_memrealtime() - t0 < DEADLINE_TICKS);
        if (!((unsigned)__builtin_amdgcn_readfirstlane(am[0] & am[1] & am[2] & am[3]) >> 31)) { __hip_atomic_store(tmo, call, AGT); return; }
        const int be = block_exp(__builtin_amdgcn_readfirstlane(max(max(am[0], am[1]), max(am[2], am[3]))) & 0x7fffffffu);
        WAIT_V(0);
        if (be) {
          unit_store<PUSH_PACE>(va, pow2f(-be), rp, po, OOB, 0, lane);
          unit_store<PUSH_PACE>(vb, pow2f(-be), rp, po + 4096, OOB, 0, lane);
          WAIT_V(0);
        }
        if (lane == 0 && __hip_atomic_fetch_add((LAS unsigned*)smem + 8, 1u, __ATOMIC_RELAXED, __HIP_MEMORY_SCOPE_WORKGROUP) == 7 - CW)
          __hip_atomic_store((GAS unsigned*)(pws + flag_off(XPAN + pb_, rank * 8, call)), tag_of(call) * 0x01010101u, SYS);
      }
      const u32x4 ro = make_rsrc(P.out, MD * ND * 4);
      const unsigned dbase = OFF_DATA + (call & 1) * PAR_BYTES + p * 8 * STEP_BYTES;
      bool dead = false;
      unsigned gates = 0;
      for (unsigned i = w - CW;; i += 4) {
        for (;;) {
          const u32x4 pr = *(volatile LAS u32x4*)smem;
          const unsigned all = __builtin_amdgcn_readfirstlane(pr[0] | pr[1] | pr[2] | pr[3]);
          const unsigned got = __builtin_amdgcn_readfirstlane(min(min(pr[0], pr[1]), min(pr[2], pr[3])));
          if (all >> 31) { dead = true; break; }
          if (got > i) break;
          if (__builtin_amdgcn_s_memrealtime() - t0 > 2 * DEADLINE_TICKS) { dead = true; __hip_atomic_store(tmo, call, AGT); break; }
          __builtin_amdgcn_s_sleep(1);
        }
        const unsigned sq = __builtin_amdgcn_readfirstlane(ctl[32 + (i & 31)]), gs = sq & 0xffffu;
        if ((sq >> 16) != i + 1 || gs >= ntot) break;
        if (dead) {
          const u32x4 nn = {0x7fc00000u, 0x7fc00000u, 0x7fc00000u, 0x7fc00000u};
          ST128(nn, ro, lv, gs >= (unsigned)NSTEP ? out_off(z0 + gs - NSTEP) : OOB, 0, "sc1");
          break;
        }
        const bool look = (gates & 0xffu) != 0xffu || (gs + 96 >= (unsigned)NSTEP && gates != 0xffffu);
        unsigned gw;
        asm volatile("buffer_load_dword %0, %1, %2, %3 offen sc0 sc1" : "=&v"(gw) : "v"(gate_off), "s"(rd), "s"(look ? 0u : OOB) : "memory");
        u32x4 d[8];
#pragma unroll
        for (int k = 0; k < 8; ++k) d[k] = *(volatile LAS u32x4*)(smem + RING + (i % NSLOT) * 8192 + k * 1024 + lv);
        const bool in = gs < (unsigned)NSTEP;
        const unsigned e = __builtin_amdgcn_readfirstlane(tab[in ? gs : 0]), row = (e >> 16) & 31;
        const u32x4 rt = in ? rd : ro;
        const unsigned o = in ? dbase + (e & 0x1fffu) * 8192 : out_off(z0 + gs - NSTEP), st = in ? 1024u : (unsigned)(ND * 4);
#pragma unroll
        for (int k = 0; k < 8; ++k) ST128(d[k], rt, lv, o + k * st, 0, "sc0 sc1");
        ctl[w] = (i >> 2) + 1;
        asm volatile("s_waitcnt vmcnt(0)" : "+v"(gw) :: "memory");
        if (in && lane == 0) {
          GAS char* f = ws + flag_off(row, p * 8 + (e >> 24), call);
          if (e & 0x8000u) __hip_atomic_store((GAS unsigned char*)(f + ((e >> 21) & 7)), (unsigned char)tag_of(call), SYS);
          else if (__hip_atomic_fetch_add((GAS unsigned*)(ws + OFF_CNT + row * FSTR + (p * 8 + (e >> 24)) * 4), 1u, AGT) + 1 == call * (row < XPAN - 1 ? 5u : 4u))
          { __hip_atomic_store((GAS unsigned char*)(f + 4), (unsigned char)tag_of(call), SYS); __hip_atomic_store((GAS unsigned*)f, tag_of(call) * 0x01010101u, SYS); }
        }
        if (look) { gates = (unsigned)__ballot(gw >= call * gate_mul) & 0xffffu; ctl[9] = gates; }
      }
      return;
    }
    CALL_ISSUE;
    if (w == 0) ctl[lane] = (lane >= 32 && lane < 32 + AHEAD) ? (lane - 31) << 16 | (q + (lane - 32) * NCOMM) : 0u;
    WAIT_L0;
    BARRIER;
    unsigned four;
    asm volatile("global_load_dword %0, %1, off" : "=v"(four) : "v"(karg + offsetof(Params, order) + ((rank - p - 1) & 7) * 256 + lane * 4) : "memory");
    const unsigned call = CALL_WAIT(0);
    asm volatile("" : "+v"(four));
    {
      unsigned sz = 0;
#pragma unroll
      for (int i = 0; i < 4; ++i) sz += (4 * lane + i < NBLK) ? ((((four >> (8 * i)) & 31) < XPAN - 1) ? 5u : 4u) : 0u;
      unsigned inc = sz;
#pragma unroll
      for (int d = 1; d < 64; d <<= 1) { const unsigned o = __shfl_up(inc, d); if (lane >= (unsigned)d) inc += o; }
      unsigned pos = inc - sz;
#pragma unroll
      for (int i = 0; i < 4; ++i) {
        const unsigned by = (four >> (8 * i)) & 255, row = by & 31, j = by >> 5;
        const unsigned n = (4 * lane + i < NBLK) ? (row < XPAN - 1 ? 5u : 4u) : 0u;
        const unsigned e = (j * (STEP_BYTES >> 13) + (row < XPAN ? row * 5 : (HALF_BYTES >> 13) + (row - XPAN) * 4)) | row << 16 | j << 24;
#pragma unroll
        for (unsigned k = 0; k < 5; ++k) if (k < n) tab[pos + k] = (e + k * 0x200001u) | (pos + DIRECT >= (unsigned)NSTEP ? 0x8000u : 0u);
        pos += n;
      }
    }
    const CAS unsigned long long* outtab = (const CAS unsigned long long*)(karg + offsetof(Params, outs));
    const u32x4 rsi = make_rsrc((const void*)pws, WS_USED), rso = make_rsrc((const void*)outtab[p], MD * ND * 4);
    const unsigned dbase = OFF_DATA + (call & 1) * PAR_BYTES + p * 8 * STEP_BYTES;
#define RING_LD(rs, so, ldsa) do { unsigned keep_; \
    asm volatile("s_mov_b32 %0, m0\n\ts_mov_b32 m0, %4\n\ts_nop 0\n\tbuffer_load_dwordx4 %1, %2, %3 offen sc0 sc1 lds\n\ts_mov_b32 m0, %0" \
                 : "=&s"(keep_) : "v"(lv), "s"(rs), "s"(so), "s"(ldsa) : "memory"); } while (0)
    GAS char* tkp = ws + OFF_TKT + p * 128;
    const unsigned tkb = NCOMM * AHEAD - (call - 1) * ntot;
    unsigned gates = 0;
    unsigned tk = 0;
    bool dead = false;
    for (unsigned i = 0;; ++i) {
      unsigned sq = __builtin_amdgcn_readfirstlane(ctl[32 + (i & 31)]);
      unsigned taken = __builtin_amdgcn_readfirstlane(ctl[4 + (i & 3)]);
      gates |= __builtin_amdgcn_readfirstlane(ctl[9]);
      unsigned gs = sq & 0xffffu, e = __builtin_amdgcn_readfirstlane(tab[min(gs, (unsigned)NSTEP - 1)]);
      auto gate_of = [&]() { return gs < (unsigned)NSTEP ? max(e >> 24, (unsigned)(((e >> 16) & 31) < XPAN)) : 8 + (((z0 + gs - NSTEP) / 20 - 4 * (dist - 1)) & 7); };
      auto can_go = [&]() { return (sq >> 16) == i + 1 && (gs >= ntot || (((gates >> gate_of()) & 1) && (i >> 2) < taken + NSLOT / 4)); };
      if (!can_go()) {
        WAIT_V(0);
        ctl[w] = i;
        const unsigned long long t0 = __builtin_amdgcn_s_memrealtime();
        for (unsigned spins = 0;; ++spins) {
          sq = __builtin_amdgcn_readfirstlane(ctl[32 + (i & 31)]);
          taken = __builtin_amdgcn_readfirstlane(ctl[4 + (i & 3)]);
          gs = sq & 0xffffu; e = __builtin_amdgcn_readfirstlane(tab[min(gs, (unsigned)NSTEP - 1)]);
          if ((sq >> 16) == i + 1 && gs < ntot && !((gates >> gate_of()) & 1))
            gates |= (unsigned)__ballot(__hip_atomic_load(gate_word, SYS) >= call * gate_mul) & 0xffffu;
          if (can_go()) break;
          if ((spins & 15) == 15) {
            const unsigned tm = __builtin_amdgcn_readfirstlane(__hip_atomic_load(tmo, AGT));
            if (tm == call || __builtin_amdgcn_s_memrealtime() - t0 > DEADLINE_TICKS) { dead = true; break; }
          }
          __builtin_amdgcn_s_sleep(1);
        }
      }
      if (dead || gs >= ntot) break;
      for (int n = 0; n < 64; ++n) {
        const u32x4 pr = *(volatile LAS u32x4*)smem;
        if (__builtin_amdgcn_readfirstlane(min(min(pr[0], pr[1]), min(pr[2], pr[3]))) + DRIFT >= i) break;
      }
      const bool in = gs < (unsigned)NSTEP;
      const u32x4 rs = in ? rsi : rso;
      const unsigned st = in ? 1024u : (unsigned)(ND * 4);
      const unsigned o = (in ? dbase + (e & 0x1fffu) * 8192 : out_off(z0 + gs - NSTEP)) + 2 * w * st;
      const unsigned la = RING + (i % NSLOT) * 8192 + w * 2048;
      if (w) WAIT_V(1); else WAIT_V(2);
      RING_LD(rs, o, la);
      if (w) WAIT_V(1); else WAIT_V(2);
      ctl[w] = i;
      RING_LD(rs, o + st, la + 1024);
      if (w == 0) {
        asm volatile("s_waitcnt vmcnt(2)" : "+v"(tk) :: "memory");
        if (i) ctl[32 + ((i - 1 + AHEAD) & 31)] = (i + AHEAD) << 16 | min(__builtin_amdgcn_readfirstlane(tk) + tkb, 0xffffu);
        unsigned long long ex2;
        asm volatile("s_mov_b64 %1, exec\n\ts_mov_b64 exec, 1\n\tglobal_atomic_add %0, %2, %3, off sc0\n\ts_mov_b64 exec, %1"
                     : "=&v"(tk), "=&s"(ex2) : "v"(tkp), "v"(1u) : "memory");
      }
    }
    asm volatile("s_waitcnt vmcnt(0)" : "+v"(tk) :: "memory");
    if (dead) __hip_atomic_store(tmo, call, AGT);
    ctl[w] = dead ? 0x80000000u : 0x40000000u;
    WAIT_L0;
    return;
  }

  unsigned call;
  {
    const bool gemm = q >= QG;
    constexpr unsigned NST = (QG - NCOMM) * 8 + NCOMM, PER = XPAN + WPAN;
    GAS char* peer_ws = (GAS char*)wstab[lane & 7];
    unsigned c = gemm ? blockIdx.x - QG * 8 : 192 + (q >= NCOMM ? blockIdx.x - NCOMM * 8 + NCOMM : q);
    f32x4 v[10];
    unsigned half = 0;
    CALL_ISSUE;
    for (unsigned i = 0; i < max(c / PER, (unsigned)(c < XPAN)); ++i) __builtin_amdgcn_s_sleep(STAGE_DELAY);
    unit_load(v, P, c % PER < XPAN, c / PER, c % PER < XPAN ? c % PER : c % PER - XPAN, w, lane);
    call = CALL_WAIT(10);
    for (;;) {
      const unsigned row = c % PER, isx = row < XPAN, t = rank * 8 + c / PER;
      UNIT_WAIT(v, 0);
      asm volatile("" : "+v"(v[8]), "+v"(v[9]));
      const unsigned so = OFF_DATA + (call & 1) * PAR_BYTES + t * STEP_BYTES + (isx ? row * A_BYTES : HALF_BYTES + (row - XPAN) * B_BYTES);
      const unsigned se = row < XPAN - 1 ? so + 32768 + w * 1024 : OOB;
      unit_store<0>(v, 1.f, rd, so + w * 4096, se, isx, lane);
      half ^= 32;
      ((volatile LAS unsigned*)(smem + half))[w] = wave_amax(v);
      WAIT_L0;
      WAIT_V(0);
      BARRIER;
      const u32x4 a0 = *(volatile LAS u32x4*)(smem + half), a1 = *(volatile LAS u32x4*)(smem + half + 16);
      const int be = block_exp(__builtin_amdgcn_readfirstlane(max(max(max(a0[0], a0[1]), max(a0[2], a0[3])), max(max(a1[0], a1[1]), max(a1[2], a1[3])))));
      if (be) {
        unit_store<0>(v, pow2f(-be), rd, so + w * 4096, se, isx, lane);
        WAIT_V(0);
        BARRIER;
      }
      if (w == 5 && lane < NR)
        __hip_atomic_store((GAS unsigned short*)(peer_ws + flag_off(row, t, call) + 6), (unsigned short)(tag_of(call) << 8 | (be & 255)), SYS);
      if (w == 7) {
        const unsigned gt = max(c / PER, isx), ro_ = OFF_RDY + (rank * 8 + gt) * 128;
        unsigned n = 0;
        if (gt && lane == 0) n = __hip_atomic_fetch_add((GAS unsigned*)(ws + ro_), 1u, AGT) + 1;
        const bool send = !gt || (unsigned)__builtin_amdgcn_readfirstlane(n) == call * (gt == 1 ? PER + XPAN : PER);
        if (send && lane < NR && lane != rank) __hip_atomic_fetch_add((GAS unsigned*)(peer_ws + ro_), 1u, SYS);
      }
      if (w == 6 && lane < 5)
        __hip_atomic_store((GAS unsigned char*)(ws + flag_off(row, t, call) + lane), (unsigned char)tag_of(call), SYS);
      c += NST;
      if (gemm || c >= 8 * PER) break;
      unit_load(v, P, c % PER < XPAN, c / PER, c % PER < XPAN ? c % PER : c % PER - XPAN, w, lane);
    }
    if (!gemm) return;
  }

  const unsigned il = ((q - QG) >> 2) * 2 + (g & 1), jl = (g >> 1) + 4 * ((q - QG) & 3);
  const unsigned pi = il + (il >= rank), pj = (jl + 2 * rank) & 15;
  const unsigned wr = w >> 2, wc = w & 3;
  constexpr unsigned K_NONE = 0x800u, K_BAD = 0x1800u;
  static_assert(K_NONE * STEP_BYTES == OOB, "an empty choice addresses nothing");

  unsigned long long remaining = ~0ull, ready = 0;
  const unsigned fx = w ? OOB : flag_off(pi, 0, call), fw = w ? OOB : flag_off(XPAN + pj, 0, call), ft = w ? OOB : OFF_TMO, l8 = lane * 8;
  const unsigned tag4 = tag_of(call) * 0x01010101u, m5 = pi == XPAN - 1 ? 0xff000000u : 0xff0000ffu;
  const unsigned selw = 2 * STAGE + (w ? 8u : 0u);
  typedef unsigned u32x2 __attribute__((ext_vector_type(2)));
  u32x2 pa = {0, 0}, pb = {0, 0};
#define POLL_LD(oa, ob) asm volatile("buffer_load_dwordx2 %0, %2, %3, %4 offen sc0 sc1\n\tbuffer_load_dwordx2 %1, %2, %3, %5 offen sc0 sc1" \
                                     : "=&v"(pa), "=&v"(pb) : "v"(l8), "s"(rd), "s"(oa), "s"(ob) : "memory")
#define POLL_WAIT(n) asm volatile("s_waitcnt vmcnt(" #n ")" : "+v"(pa), "+v"(pb) :: "memory")
#define POLL_ISSUE POLL_LD(remaining ? fx : ft, fw)
#define POLL_MASK __ballot((((pa[0] ^ tag4) | (pb[0] ^ tag4)) | ((pa[1] ^ tag4) & m5) | ((pb[1] ^ tag4) >> 24)) == 0)
#define POLL_EXP(k) (((unsigned)__builtin_amdgcn_readlane((int)(pa[1] + pb[1]), (k) & 63)) & 0xff0000u)
  auto poll_eval = [&]() -> unsigned {
    const unsigned none = (!remaining && (unsigned)__builtin_amdgcn_readfirstlane(pa[0]) == call) ? K_BAD : K_NONE;
    ready |= POLL_MASK;
    const unsigned long long av = ready & remaining;
    remaining &= ~(av & (0ull - av));
    const unsigned k = __builtin_ctzll(av | 1ull << 63);
    return av ? k | POLL_EXP(k) : none;
  };
  auto pick = [&]() -> unsigned {
    const unsigned long long t0 = __builtin_amdgcn_s_memrealtime();
    for (unsigned spins = 1;; ++spins) {
      POLL_LD(fx, fw); POLL_WAIT(0);
      const unsigned k = poll_eval();
      if (k != K_NONE) return k;
      if ((spins & 31) == 0) {
        POLL_LD(ft, ft); POLL_WAIT(0);
        if (__builtin_amdgcn_readfirstlane(pa[0]) == call || __builtin_amdgcn_s_memrealtime() - t0 > DEADLINE_TICKS) break;
      }
      __builtin_amdgcn_s_sleep(POLL_SLEEP);
    }
    __hip_atomic_store(tmo, call, AGT);
    return K_BAD;
  };

  const auto rw = __builtin_amdgcn_make_buffer_rsrc((void*)ws, 0, WS_USED, 0x00020000);
  const unsigned dbase = OFF_DATA + (call & 1) * PAR_BYTES;
  const unsigned vdma = w * 4096 + lane * 16;
  const unsigned vext = (pi == XPAN - 1) ? OOB : 32768 + w * 1024 + lane * 16;
  const unsigned abase = dbase + pi * A_BYTES, bbase = dbase + HALF_BYTES + pj * B_BYTES;
#define DMA_B(koff, buf, i) __builtin_amdgcn_raw_ptr_buffer_load_lds(rw, (LAS char*)smem + (buf) * STAGE + A_BYTES + w * 4096, 16, vdma, bbase + (koff), (i) * 1024, AUX_DMA)
#define DMA_A(koff, buf, i) __builtin_amdgcn_raw_ptr_buffer_load_lds(rw, (LAS char*)smem + (buf) * STAGE + w * 4096, 16, vdma, abase + (koff), (i) * 1024, AUX_DMA)
#define DMA_E(koff, buf) __builtin_amdgcn_raw_ptr_buffer_load_lds(rw, (LAS char*)smem + (buf) * STAGE + 32768 + w * 1024, 16, vext, abase + (koff), 0, AUX_DMA)
#define SCHED __builtin_amdgcn_sched_barrier(0)
#define LGKM0 __builtin_amdgcn_s_waitcnt(0xC07F)

  f32x4 acc[10][4];
#pragma unroll
  for (int m = 0; m < 10; ++m)
#pragma unroll
    for (int n = 0; n < 4; ++n) acc[m][n] = (f32x4){0.f, 0.f, 0.f, 0.f};

  const unsigned aoff = wr * 20480 + (lane & 15) * 128 + (((lane >> 4) ^ (lane >> 1)) & 7) * 16, boff = A_BYTES + wc * 8192 + lane * 16;
  h16x8 fa[2][5], fb[2][4];
#define RD_B(buf, kk, n) fb[kk][n] = *(const LAS h16x8*)((const LAS char*)smem + (buf) * STAGE + boff + (kk) * 4096 + (n) * 1024)
#define RD_A(buf, ph, m) fa[(ph) & 1][m] = *(const LAS h16x8*)((const LAS char*)smem + (buf) * STAGE + (aoff ^ (((ph) >> 1) * 64)) + (((ph) & 1) * 5 + (m)) * 2048)
#define MM(ph, m, n) acc[((ph) & 1) * 5 + (m)][n] = __builtin_amdgcn_mfma_f32_16x16x32_f16(fa[(ph) & 1][m], fb[(ph) >> 1][n], acc[((ph) & 1) * 5 + (m)][n], 0, 0, 0)
#define M1(ph, i, X) MM(ph, (i) >> 2, (i) & 3); SCHED; X; SCHED
#define PIN(x) asm volatile("" : "+s"(x))
#define PRIO(n) __builtin_amdgcn_s_setprio(n)
#define NOP_ do {} while (0)
#define SEL_PUT(k) *(volatile LAS unsigned*)(smem + selw + cur * 4) = (k)
#define SEL_GET asm volatile("ds_read_b32 %0, %1" : "=v"(ks) : "v"(2 * STAGE + cur * 4) : "memory")
#define SEL_TAKE(n) asm volatile("s_waitcnt lgkmcnt(%1)" : "+v"(ks) : "n"(n) : "memory"); k2 = __builtin_amdgcn_readfirstlane(ks); kn = (k2 & 0xfffu) * STEP_BYTES
  unsigned kn = OOB, cur = 0, cv = 0, nv = 0, ndone = 0, bad = 0, go;
  unsigned ea = 0, ec = 0, en = 0;
#define RESCALE \
    if (ec != ea) {                  \
      const float f = pow2f((int)(signed char)ea - (int)(signed char)ec); \
      _Pragma("unroll") for (int m = 0; m < 10; ++m) _Pragma("unroll") for (int n = 0; n < 4; ++n) acc[m][n] *= f; \
      ea = ec; \
    }
#define REFILL \
    while (!cv) {                    \
      const unsigned nxt = cur ^ 1; \
      unsigned ks, k2; \
      DMA_A(kn, nxt, 0); DMA_A(kn, nxt, 1); DMA_A(kn, nxt, 2); DMA_A(kn, nxt, 3); \
      POLL_ISSUE; POLL_WAIT(0); \
      unsigned k = poll_eval(); \
      if (w == 0 && k == K_NONE && !nv) k = pick(); \
      SEL_PUT(k); \
      LGKM0; BARRIER; \
      SEL_GET; SEL_TAKE(0); \
      DMA_B(kn, cur, 0); DMA_B(kn, cur, 1); DMA_B(kn, cur, 2); DMA_B(kn, cur, 3); DMA_E(kn, cur); \
      bad = (k2 >> 12) & 1; ec = en; en = k2 >> 16; \
      cv = nv | bad; nv = (~k2 >> 11) & 1; cur = nxt;     \
    }
  do {
    REFILL;
    RESCALE;
    RD_B(cur, 0, 0); RD_A(cur, 0, 0); RD_B(cur, 0, 1); RD_B(cur, 0, 2); RD_B(cur, 0, 3);
    RD_A(cur, 0, 1); RD_A(cur, 0, 2); RD_A(cur, 0, 3); RD_A(cur, 0, 4);
    do {
      const unsigned nxt = cur ^ 1;
      unsigned k, ks, k2;
      PRIO(2); LGKM0; SCHED;
      M1(0, 0, RD_A(cur, 1, 0)); M1(0, 1, RD_A(cur, 1, 1)); M1(0, 2, RD_A(cur, 1, 2)); M1(0, 3, RD_A(cur, 1, 3); DMA_A(kn, nxt, 0));
      M1(0, 4, RD_A(cur, 1, 4)); M1(0, 5, NOP_); M1(0, 6, NOP_); M1(0, 7, DMA_A(kn, nxt, 1));
      M1(0, 8, RD_B(cur, 1, 0)); M1(0, 9, RD_B(cur, 1, 1)); M1(0, 10, NOP_); M1(0, 11, DMA_A(kn, nxt, 2));
      M1(0, 12, NOP_); M1(0, 13, NOP_); M1(0, 14, NOP_); M1(0, 15, DMA_A(kn, nxt, 3));
      M1(0, 16, NOP_); M1(0, 17, NOP_); M1(0, 18, POLL_ISSUE); M1(0, 19, NOP_);
      PRIO(1); LGKM0; SCHED;
      M1(1, 0, RD_A(cur, 2, 0)); M1(1, 1, RD_A(cur, 2, 1)); M1(1, 2, RD_A(cur, 2, 2)); M1(1, 3, RD_A(cur, 2, 3));
      M1(1, 4, RD_A(cur, 2, 4)); M1(1, 5, NOP_); M1(1, 6, NOP_); M1(1, 7, NOP_);
      M1(1, 8, RD_B(cur, 1, 2)); M1(1, 9, RD_B(cur, 1, 3)); M1(1, 10, NOP_); M1(1, 11, NOP_);
      M1(1, 12, NOP_); M1(1, 13, NOP_); M1(1, 14, NOP_); M1(1, 15, NOP_);
      M1(1, 16, NOP_); M1(1, 17, NOP_); M1(1, 18, NOP_); M1(1, 19, NOP_);
      PRIO(0); LGKM0; SCHED;
      M1(2, 0, RD_A(cur, 3, 0)); M1(2, 1, RD_A(cur, 3, 1)); M1(2, 2, RD_A(cur, 3, 2)); M1(2, 3, RD_A(cur, 3, 3));
      M1(2, 4, RD_A(cur, 3, 4)); M1(2, 5, NOP_); M1(2, 6, NOP_); M1(2, 7, NOP_);
      M1(2, 8, NOP_); M1(2, 9, NOP_); M1(2, 10, NOP_); M1(2, 11, POLL_WAIT(0));
      unsigned long long av;
      M1(2, 12, k = __builtin_amdgcn_readfirstlane(pa[0]); av = POLL_MASK; PIN(k); PIN(av));
      M1(2, 13, k = k == call ? K_BAD : K_NONE; k = remaining ? K_NONE : k; PIN(k));
      M1(2, 14, ready |= av; av = ready & remaining; k = av ? (unsigned)__builtin_ctzll(av) : k; PIN(k); PIN(av));
      M1(2, 15, k |= POLL_EXP(k); SEL_PUT(k));
      M1(2, 16, remaining &= ~(av & (0ull - av)); PIN(remaining)); M1(2, 17, NOP_); M1(2, 18, NOP_); M1(2, 19, NOP_);
      PRIO(3); LGKM0; BARRIER;
      SEL_GET; SCHED;
      M1(3, 0, RD_B(nxt, 0, 0)); M1(3, 1, RD_A(nxt, 0, 0)); M1(3, 2, RD_B(nxt, 0, 1)); M1(3, 3, RD_B(nxt, 0, 2));
      M1(3, 4, SEL_TAKE(4); PIN(kn)); M1(3, 5, DMA_B(kn, cur, 0); RD_B(nxt, 0, 3)); M1(3, 6, RD_A(nxt, 0, 1)); M1(3, 7, RD_A(nxt, 0, 2));
      M1(3, 8, DMA_B(kn, cur, 1); RD_A(nxt, 0, 3)); M1(3, 9, RD_A(nxt, 0, 4)); M1(3, 10, bad |= (k2 >> 12) & 1; PIN(bad)); M1(3, 11, DMA_B(kn, cur, 2));
      M1(3, 12, ndone += 1 + bad * KS; PIN(ndone)); M1(3, 13, cv = nv | bad; go = nv & ((ndone - (KS - 1)) >> 31); PIN(cv); PIN(go)); M1(3, 14, nv = (~k2 >> 11) & 1; PIN(nv)); M1(3, 15, DMA_B(kn, cur, 3));
      M1(3, 16, ec = en; en = k2 >> 16; go &= ((ec ^ ea) - 1) >> 31; PIN(ec); PIN(en); PIN(go)); M1(3, 17, NOP_); M1(3, 18, DMA_E(kn, cur)); M1(3, 19, NOP_);
      cur = nxt;
    } while (go);
  } while (ndone < (unsigned)KS - 1);
  REFILL;
  RESCALE;

  const u32x4 ro = make_rsrc(P.out, MD * ND * 4);
  unsigned lz = l8;
  asm volatile("" : "+v"(lz));
  const unsigned ov = (lz >> 7) * 4 * (ND * 4) + (lz & 120) * 2;
  const unsigned os = (pi * 320 + wr * 160) * (ND * 4) + (pj * 256 + wc * 64) * 4;
  const float osc = bad ? __builtin_nanf("") : pow2f((int)(signed char)ea);
  const bool exported = pi < (unsigned)NR && jl < 4;
  if (exported) {
    if (threadIdx.x < 2) *(volatile LAS unsigned*)(smem + 2 * STAGE + 16 + threadIdx.x * 4) = 0;
    if (wr) BARRIER;
  }
  PRIO(0);
  h16x8 ga[10][2];
#define RD_L(m) do { ga[m][0] = *(const LAS h16x8*)((const LAS char*)smem + cur * STAGE + aoff + (m) * 2048); \
                     ga[m][1] = *(const LAS h16x8*)((const LAS char*)smem + cur * STAGE + (aoff ^ 64) + (m) * 2048); } while (0)
#define MM_L(m) do { _Pragma("unroll") for (int kk = 0; kk < 2; ++kk) _Pragma("unroll") for (int n = 0; n < 4; ++n) \
                       acc[m][n] = __builtin_amdgcn_mfma_f32_16x16x32_f16(ga[m][kk], fb[kk][n], acc[m][n], 0, 0, 0); } while (0)
  RD_L(0);
#pragma unroll
  for (int n = 0; n < 4; ++n) { RD_B(cur, 0, n); RD_B(cur, 1, n); }
  RD_L(1);
  SCHED;
  MM_L(0);
#pragma unroll
  for (int m = 0; m < 10; ++m) {
    SCHED;
    if (m + 2 < 10) RD_L(m + 2 < 10 ? m + 2 : 0);
    if (m + 1 < 10) MM_L(m + 1 < 10 ? m + 1 : 0);
    SCHED;
#pragma unroll
    for (int r = 0; r < 4; ++r) {
      const f32x4 o = {acc[m][0][r] * osc, acc[m][1][r] * osc, acc[m][2][r] * osc, acc[m][3][r] * osc};
      ST128(o, ro, ov, os + (m * 16 + r) * (ND * 4), 0, "sc0 sc1");
    }
  }
  if (exported) {
    if (!wr) { WAIT_L0; BARRIER; }
    WAIT_V(0);
    if (lane == 0 && __hip_atomic_fetch_add((LAS unsigned*)(smem + 2 * STAGE + 16) + wr, 1u, __ATOMIC_RELAXED, __HIP_MEMORY_SCOPE_WORKGROUP) == 3)
      __hip_atomic_store((GAS unsigned*)((GAS char*)wstab[pi] + OFF_DONE + (rank * 8 + jl * 2 + wr) * 128), call, SYS);
  }
}

static const unsigned char ORDER[7][224] = {
  {13,15,24,14,17,12,21,23,22,25,26,19,16,20,27,18,0,6,4,11,3,5,2,7,1,9,10,8,47,56,45,46,36,32,49,44,38,54,34,53,37,42,35,33,51,41,57,40,48,52,39,43,59,50,55,58,77,68,64,85,76,66,81,69,72,70,67,78,84,88,65,80,79,73,91,83,71,74,82,75,86,89,90,87,109,98,100,108,110,99,96,97,102,115,101,112,113,111,106,122,118,123,105,107,103,120,121,116,119,104,114,141,134,140,129,117,131,142,130,133,132,128,145,143,149,146,148,138,151,150,139,147,155,136,164,160,181,174,161,176,204,236,196,178,205,173,182,197,166,184,153,206,172,192,152,162,135,163,186,165,171,154,137,183,170,224,187,144,168,177,228,179,169,198,237,207,185,194,225,193,175,180,195,208,230,210,227,209,229,167,215,238,217,214,212,201,218,239,226,200,219,199,211,247,216,231,246,213,203,202,249,245,248,232,242,235,240,244,251,234,241,250,243,233},
  {26,15,12,27,22,17,13,24,20,14,18,21,25,19,16,23,1,7,10,8,6,5,2,3,9,0,11,4,44,58,33,42,47,54,59,40,52,38,46,34,41,53,37,35,51,45,39,57,49,36,43,32,48,50,56,55,76,86,65,72,66,73,77,79,70,78,69,85,64,91,90,83,67,68,75,80,81,82,84,89,88,74,87,71,109,101,102,108,110,99,98,100,96,97,116,119,120,115,118,121,107,117,103,114,105,111,104,112,113,106,123,140,131,141,133,128,142,134,122,132,130,129,146,148,149,144,139,155,136,152,137,154,153,135,182,174,179,165,164,184,180,150,169,175,181,212,172,160,237,205,185,178,162,196,177,238,145,168,173,176,186,206,161,183,171,163,166,151,218,147,201,197,187,236,138,143,192,204,229,167,195,199,194,215,226,217,198,193,211,209,170,225,207,214,219,228,208,210,227,203,230,224,241,216,213,240,244,202,233,200,239,247,250,232,243,246,251,248,242,249,235,245,234,231},
  {19,23,12,13,20,21,14,16,27,15,22,18,17,25,24,26,5,1,10,8,6,2,0,9,4,3,7,11,51,33,34,32,40,45,44,37,54,46,41,38,59,36,48,35,50,52,47,43,49,56,55,42,39,53,57,58,65,86,76,68,64,77,80,78,67,66,69,89,79,87,70,74,88,71,90,85,73,84,81,72,75,83,82,91,96,99,108,109,101,98,111,102,97,110,100,117,116,120,118,122,105,134,141,129,140,133,142,128,131,132,130,143,112,106,114,123,103,121,113,115,119,107,104,145,172,163,173,164,152,146,144,148,136,155,201,195,135,196,179,162,197,211,176,175,147,160,212,210,161,153,209,200,178,219,180,207,139,198,205,204,170,192,186,218,138,185,166,194,151,193,214,183,167,149,165,168,177,174,208,169,216,202,217,206,181,137,182,213,187,154,184,215,237,203,236,224,225,227,229,230,226,199,228,238,150,242,243,250,241,248,247,171,233,239,251,232,234,240,231,245,246,249,244,235},
  {12,13,26,19,14,17,15,27,20,18,24,23,16,22,21,25,5,6,1,4,3,11,2,0,9,8,7,10,44,45,51,46,36,43,33,38,37,47,35,52,58,50,55,32,34,40,59,57,42,49,54,56,41,39,48,53,76,83,78,77,69,68,70,64,84,65,67,66,75,87,80,81,72,82,89,74,73,79,88,86,71,91,85,90,97,108,109,102,100,110,101,99,113,96,98,114,111,112,106,104,119,122,120,121,107,140,133,141,134,132,129,142,115,103,130,128,118,143,116,123,117,105,131,144,150,153,147,137,154,145,135,138,149,151,152,177,175,170,187,183,171,174,160,186,185,165,173,172,167,155,161,206,181,196,139,228,162,166,237,204,136,163,148,236,179,176,218,169,168,164,198,184,180,207,195,178,205,194,224,146,192,230,193,225,229,182,197,216,227,208,213,226,219,210,201,209,215,214,217,202,211,245,200,212,203,239,238,249,199,251,246,233,244,232,242,248,231,250,241,247,240,243,234,235},
  {12,15,20,17,14,27,23,16,13,21,19,24,22,25,26,18,5,6,2,3,4,11,7,1,10,0,8,9,47,52,44,35,49,38,46,37,48,36,59,45,55,33,54,43,40,58,53,34,39,50,32,41,42,51,57,56,76,77,69,67,66,78,65,87,72,68,70,82,79,64,81,71,85,89,84,86,90,75,80,83,91,74,88,73,101,110,108,100,109,98,96,99,111,102,97,123,107,118,114,104,116,112,121,115,106,117,105,141,132,140,130,128,134,120,113,142,119,129,131,143,133,103,122,164,172,173,161,174,163,166,162,165,160,175,225,178,177,147,137,181,187,204,136,179,195,154,148,227,198,146,138,176,183,212,151,237,139,230,144,167,192,236,193,205,149,182,197,238,169,180,135,150,155,226,206,168,171,153,185,228,229,186,184,207,170,152,216,219,196,194,145,211,224,210,215,203,208,199,214,213,218,202,201,245,209,217,200,242,249,244,251,246,240,234,231,239,241,233,243,250,232,247,235,248},
  {14,25,23,12,16,17,13,15,27,18,24,26,21,19,22,20,1,2,4,6,7,5,0,8,3,10,11,9,46,33,57,36,38,44,32,45,37,34,42,56,58,35,49,50,53,41,55,47,48,39,43,59,51,54,52,40,64,78,77,65,88,87,66,70,90,69,76,79,85,74,71,68,67,73,75,82,80,83,86,81,91,72,84,89,109,108,96,98,97,101,117,100,99,110,113,102,119,122,114,111,105,107,112,123,140,130,128,141,129,131,142,133,132,103,121,116,115,120,104,118,134,106,172,160,173,166,165,174,161,143,162,164,175,163,192,202,171,210,177,194,193,152,187,176,149,180,139,178,137,151,212,207,145,182,205,186,167,206,155,179,170,204,199,226,153,135,147,197,236,224,184,148,195,196,228,181,198,214,169,215,150,208,237,217,230,218,219,144,213,238,200,203,227,136,185,168,216,138,225,211,209,229,154,201,146,183,239,251,240,250,242,231,243,241,249,235,247,245,244,232,248,246,233,234},
  {26,18,16,25,12,24,14,13,27,19,15,21,22,23,20,17,3,4,2,0,5,7,10,1,6,11,9,8,50,35,57,34,37,58,32,51,47,46,43,45,55,59,52,44,33,39,36,42,38,53,41,49,56,40,54,48,82,67,69,90,77,78,64,76,84,68,65,70,79,66,72,83,85,75,87,91,71,89,73,74,88,80,81,86,97,102,108,109,110,96,100,111,101,99,98,115,117,103,116,113,104,134,141,140,142,132,130,131,143,129,128,133,118,122,107,119,112,114,105,106,120,121,123,147,151,172,162,173,160,174,165,161,150,146,163,193,205,168,213,149,202,195,216,154,209,204,184,198,138,185,215,196,169,175,194,214,166,137,176,170,136,218,139,152,199,197,182,192,153,206,135,207,148,178,177,183,186,208,171,187,145,155,203,180,164,181,179,200,167,211,237,228,217,210,236,230,226,238,229,224,225,227,201,212,144,219,248,247,239,240,243,241,250,234,245,244,249,246,235,233,251,231,242,232}
};

static void fill_order(unsigned char (*order)[256], unsigned rank) {
  for (unsigned lk = 0; lk < NR - 1; ++lk) {
    unsigned n = 0;
    for (unsigned s = 0; s < 8 * (XPAN - 1 + WPAN); ++s) {
      const unsigned by = ORDER[lk][s], id = by & 31, j = by >> 5;
      const unsigned row = id < 12 ? id + (id >= rank) : XPAN + ((id - 12 + 2 * rank) & 15);
      bool pushed = false;
      for (unsigned i = 0; i < (unsigned)(NCOMM * PUSHN); ++i) pushed |= (j == 0 && row == XPAN + push_panel(lk, i));
      if (!pushed) order[lk][n++] = (unsigned char)(j << 5 | row);
    }
    while (n < 256) order[lk][n++] = 0;
  }
}

extern "C" void kernel_launch(int rank, int nranks, ncclComm_t comm,
                              hipStream_t stream,
                              void* const* d_in, const int* in_sizes, int n_in,
                              void* d_out, int out_size,
                              const int* in_dtypes, int out_dtype,
                              void* const* d_out_all,
                              void* const* d_ws_all, size_t ws_bytes) {
  if (nranks != NR || (unsigned)rank >= (unsigned)NR || n_in != 2 || in_sizes[0] != MD * KL || in_sizes[1] != KL * ND ||
      out_size != MD * ND || ws_bytes < (size_t)WS_USED) {
    fprintf(stderr, "unexpected problem shape\n");
    return;
  }
  Params P;
  P.x = (const float*)d_in[0]; P.w = (const float*)d_in[1]; P.out = (float*)d_out;
  for (int i = 0; i < NR; ++i) { P.ws[i] = (char*)d_ws_all[i]; P.outs[i] = (char*)d_out_all[i]; }
  P.rank = rank; P.pad = 0;
  fill_order(P.order, (unsigned)rank);
  HIP_CHECK(hipFuncSetAttribute((const void*)fused_gemm_ar, hipFuncAttributeMaxDynamicSharedMemorySize, LDS_BYTES));
  fused_gemm_ar<<<GRID, BLOCK, LDS_BYTES, stream>>>(P);
  HIP_CHECK(hipGetLastError());
}
```
